```python
import numpy as np

kernel_name = "gpu_a2a_gemm_m4096_k4096_n4096_w8a8_dqsilu"


ABSMAX_RTOL = 0.001

COMPUTE_DTYPE = "int8"
EPILOGUE = "dequant_silu"
IN_QUANT = "int8"
OUT_QUANT = None
IN_DTYPES = {"x": "int8", "w_mat": "int8",
             "scale_x": "float32", "scale_w": "float32"}
WORLD_SIZE = 8

def setup_inputs():

    x = np.random.randint(-127, 128, size=(4096, 4096)).astype(np.int8)
    w_mat = np.random.randint(-127, 128, size=(4096, 4096)).astype(np.int8)
    scale_x = (0.02 * (1.0 + np.random.rand(1))).astype(np.float32)
    scale_w = (0.02 * (1.0 + np.random.rand(1))).astype(np.float32)
    return {"x": x, "w_mat": w_mat, "scale_x": scale_x, "scale_w": scale_w}

def reference(x, w_mat, scale_x, scale_w):
    acc = x.astype(np.float64) @ w_mat.astype(np.float64)
    y = (acc.astype(np.float32) * (scale_x * scale_w)).astype(np.float32)
    return (y / (1.0 + np.exp(-np.clip(y, -60.0, 60.0)))).astype(np.float32)
```

```cpp
#include <hip/hip_runtime.h>
#include <rccl/rccl.h>
#include <cstdio>
#include <stdint.h>

#define HIP_CHECK(expr) do { hipError_t _e = (expr); if (_e != hipSuccess) { \
    fprintf(stderr, "HIP error %s at %s:%d\n", hipGetErrorString(_e), __FILE__, __LINE__); \
    return; } } while (0)

#define LAS __attribute__((address_space(3)))
#define GAS __attribute__((address_space(1)))
typedef int v2i __attribute__((ext_vector_type(2)));
typedef int v4i __attribute__((ext_vector_type(4)));
typedef float v4f __attribute__((ext_vector_type(4)));
typedef unsigned v4u __attribute__((ext_vector_type(4)));
typedef int v16i __attribute__((ext_vector_type(16)));

constexpr int NR = 8, KTOT = 4096, NTOT = 4096, MLOC = 512, KC = 512;
constexpr unsigned A_PAR_BYTES = 2u << 20;
constexpr unsigned FLAG_OFF = 4u << 20;
constexpr unsigned CTR_OFF = FLAG_OFF + (64u << 10);
constexpr unsigned TMO_OFF = FLAG_OFF + (128u << 10);
constexpr unsigned DBG_OFF = 8u << 20;
constexpr size_t WS_NEEDED = (size_t)(9u << 20);
constexpr unsigned long long DEADLINE_TICKS = 50000000ull;

constexpr int NS = 8;
constexpr int SLOT = 8192;
constexpr int RING_BYTES = 2 * NS * SLOT;
constexpr int LDS_BYTES = RING_BYTES + 512;
constexpr int XROW = 528;
#define INFL 1
#define TAU 70
#define T0 180
#define DN 4
#define PEEL 0
#define DRY 1
#define WWARM 1
#define PUSHFIRST 1
#define WT0 60
#define WTAU 50
#define HPRIO 3
#define A_AUX 17
#define W_AUX 0
#define EARLYWARM 1
#define RUNWAY 7
#define PSLEEP 10
constexpr int CW_FULL = 0, CW_DONE = 4, CW_READY = 8, CW_POISON = 9, CW_XRDY = 12;

struct Params {
    const int8_t* x; const int8_t* w; const float* sx; const float* sw; float* out;
    char* wsl; char* ws[NR]; int rank;
};

template <int N> __device__ __forceinline__ void wait_vm() { asm volatile("s_waitcnt vmcnt(%0)" :: "n"(N) : "memory"); }
__device__ __forceinline__ void wait_lgkm0() { asm volatile("s_waitcnt lgkmcnt(0)" ::: "memory"); }
__device__ __forceinline__ unsigned long long rt() { return __builtin_amdgcn_s_memrealtime(); }

#define NMAP 0
__device__ __forceinline__ int ntile_of(int xcd, int nl) {
    return NMAP == 0 ? xcd * 4 + nl : NMAP == 1 ? nl * 8 + xcd : NMAP == 2 ? (xcd & 3) + 4 * nl + 16 * (xcd >> 2) : (xcd & 1) + 2 * nl + 8 * (xcd >> 1);
}

__device__ __forceinline__ int kblk(int i, int rank) {
    if (i < 8) return rank * 8 + i;
    const int j = i - 8, s = j / 7;
    return (((rank + 1 + (j - 7 * s)) & 7) << 3) + s;
}

#define STAMP(slot) do {} while (0)
#define CSTAMP(slot) do {} while (0)

__global__ void __launch_bounds__(512, 2) a2a_gemm_kernel(Params P) {
    __shared__ __attribute__((aligned(16))) unsigned char lds[LDS_BYTES];
    LAS unsigned char* L = (LAS unsigned char*)lds;
    volatile LAS unsigned* CW = (volatile LAS unsigned*)(L + RING_BYTES);
    const int tid = threadIdx.x, lane = tid & 63;
    const int wave = __builtin_amdgcn_readfirstlane(tid >> 6);
    const int b = blockIdx.x;
    const int xcd = b & 7, idx = b >> 3;
    const int mt = idx & 7, ntile = ntile_of(xcd, idx >> 3);
    const int n0 = ntile * 128;
    const int rank = P.rank;
    char* wsl = P.wsl;
    unsigned* ctrp = (unsigned*)(wsl + CTR_OFF + b * 64 + wave * 4);

    float sxv, swv;
    if (wave >= 4) __builtin_amdgcn_s_setprio(HPRIO);
    if (wave == 4 || wave == 5) {
        const int ks = wave - 4;
        const int c0 = (lane & 7) ^ ((lane >> 4) & 3);
        const unsigned vo0 = (unsigned)(lane >> 3) * NTOT + (unsigned)c0 * 16u;
        const unsigned vo1 = (unsigned)(lane >> 3) * NTOT + (unsigned)(c0 ^ 4) * 16u;
        const __amdgpu_buffer_rsrc_t rsW = __builtin_amdgcn_make_buffer_rsrc((void*)P.w, 0, KTOT * NTOT, 0x00020000);
        unsigned char* ring = lds + ks * (NS * SLOT);
        __builtin_amdgcn_s_barrier();
        unsigned dmin = 0;
        volatile LAS unsigned* dp = CW + CW_DONE + 2 * ks;
        int pub = 0;
        volatile LAS unsigned* fullp = CW + CW_FULL + ks;
#pragma unroll 1
        for (int t = 0; t < 32; ++t) {
            if (__builtin_expect(dmin + 7u < (unsigned)t, 0)) {
                do {
                    if (pub < t) {
                        const int o = t - pub;
                        if (o == 1) wait_vm<0>(); else if (o == 2) wait_vm<8>(); else if (o == 3) wait_vm<16>(); else wait_vm<24>();
                        ++pub; *fullp = (unsigned)pub;
                    }
                    const unsigned d0 = dp[0], d1 = dp[1];
                    dmin = __builtin_amdgcn_readfirstlane(d0 < d1 ? d0 : d1);
                } while (dmin + 7u < (unsigned)t);
            }
            asm volatile("" ::: "memory");
            const unsigned e0 = dp[0], e1 = dp[1];
            const unsigned so = (unsigned)kblk(2 * t + ks, rank) * (64u * NTOT) + (unsigned)n0;
            unsigned char* dst = ring + (t & 7) * SLOT;
#pragma unroll
            for (int j = 0; j < 8; ++j)
                __builtin_amdgcn_raw_ptr_buffer_load_lds(rsW, (LAS void*)(dst + j * 1024), 16, (j & 2) ? vo1 : vo0, so + (unsigned)j * (8u * NTOT), 0, W_AUX);
            wait_vm<8 * INFL>();
            if (t + 1 - INFL > pub) { pub = t + 1 - INFL; *fullp = (unsigned)pub; }
            asm volatile("" ::: "memory");
            dmin = __builtin_amdgcn_readfirstlane(e0 < e1 ? e0 : e1);
        }
        if (INFL >= 4 && pub < 29) { wait_vm<24>(); *fullp = 29u; }
        if (INFL >= 3 && pub < 30) { wait_vm<16>(); *fullp = 30u; }
        if (INFL >= 2 && pub < 31) { wait_vm<8>(); *fullp = 31u; }
        wait_vm<0>();
        *fullp = 32u;
        sxv = P.sx[0]; swv = P.sw[0];
    } else if (wave == 6) {
        const int q = b & 7, rg = (b >> 3) & 3, st = b >> 5;
        const unsigned long long tstart = rt();
        v4i f[8];
        const int8_t* src = P.x + (size_t)(q * MLOC + rg * 128 + (lane & 15)) * KC + st * 64 + (lane >> 4) * 16;
        unsigned call;
#pragma unroll
        for (int i = 0; i < 8; ++i) {
            const int8_t* a = src + i * 16 * KC;
            asm volatile("global_load_dwordx4 %0, %1, off" : "=&v"(f[i]) : "v"(a) : "memory");
        }
        asm volatile("global_load_dword %0, %1, off sc1" : "=&v"(call) : "v"(ctrp) : "memory");
        asm volatile("global_load_dword %0, %1, off" : "=&v"(sxv) : "v"(P.sx) : "memory");
        asm volatile("global_load_dword %0, %1, off" : "=&v"(swv) : "v"(P.sw) : "memory");
        auto warm = [&](int e, int t0w, int tauw) {
            { const unsigned long long wdue = (unsigned long long)(t0w + e * tauw); while (rt() - tstart < wdue) __builtin_amdgcn_s_sleep(2); }
            const int8_t* wwp = P.w + (size_t)(lane >> 2) * NTOT + ntile_of(xcd, lane & 3) * 128;
#pragma unroll
            for (int i = 0; i < 8; ++i) {
                const int id = rg * 8 + i;
                const int8_t* a = wwp + ((size_t)kblk(8 * e + (id >> 2), rank) * 64 + (id & 3) * 16) * NTOT;
                unsigned keep;
                asm volatile("s_mov_b32 %0, m0\n\ts_mov_b32 m0, %2\n\ts_nop 0\n\tglobal_load_lds_dword %1, off\n\ts_mov_b32 m0, %0"
                             : "=&s"(keep) : "v"(a), "s"((unsigned)(RING_BYTES + 256)) : "memory");
            }
        };
        asm volatile("" ::: "memory");
        if (st >= 4) { warm((st + 4) & 7, WT0, WTAU); wait_vm<8>(); } else wait_vm<0>();
        asm volatile("" : "+v"(call), "+v"(sxv), "+v"(swv));
#pragma unroll
        for (int i = 0; i < 8; ++i) asm volatile("" : "+v"(f[i]));
        __builtin_amdgcn_s_barrier();
        call = __builtin_amdgcn_readfirstlane(call) + 1u;
        if (q != rank) {
            char* pw = P.ws[q];
            char* dst = pw + (call & 1u) * A_PAR_BYTES + (unsigned)(rg * 8 * 64 + st * 7 + ((rank - q - 1) & 7)) * 1024u + lane * 16;
            unsigned* fl = (unsigned*)(pw + FLAG_OFF) + (rg * 8 + st) * 32 + rank;
            if (st > 0) {
                const unsigned long long due = (unsigned long long)(T0 + st * TAU);
                while (rt() - tstart + 30 < due) __builtin_amdgcn_s_sleep(8);
                while (rt() - tstart < due) __builtin_amdgcn_s_sleep(1);
            }
#pragma unroll
            for (int i = 0; i < 8; ++i) {
                char* p = dst + i * 65536;
                asm volatile("global_store_dwordx4 %0, %1, off sc0 sc1\n\ts_nop 1" :: "v"(p), "v"(f[i]) : "memory");
            }
            wait_vm<0>();
            if (lane == 0) __hip_atomic_store(fl, call, __ATOMIC_RELAXED, __HIP_MEMORY_SCOPE_SYSTEM);
            STAMP(16 + st);
        }
        if (lane == 0) __hip_atomic_store(ctrp, call, __ATOMIC_RELAXED, __HIP_MEMORY_SCOPE_AGENT);
        if (st < 4) warm((st + 4) & 7, WT0, WTAU);
        wait_vm<0>();
    } else if (wave == 7) {
        unsigned call = __hip_atomic_load(ctrp, __ATOMIC_RELAXED, __HIP_MEMORY_SCOPE_AGENT);
        sxv = P.sx[0]; swv = P.sw[0];
        if (lane < 16) CW[lane] = 0u;
        wait_lgkm0();
        __builtin_amdgcn_s_barrier();
        call = __builtin_amdgcn_readfirstlane(call) + 1u;
        if (lane == 0) __hip_atomic_store(ctrp, call, __ATOMIC_RELAXED, __HIP_MEMORY_SCOPE_AGENT);
        const char* fp = wsl + FLAG_OFF + (mt >> 1) * 1024 + ((lane >> 3) & 1) * 128 + (lane & 7) * 4;
        unsigned* tmo = (unsigned*)(wsl + TMO_OFF);
        const bool self = (lane & 7) == rank;
        const unsigned long long t0 = rt();
        int rdy = 0;
        for (int it = 0;; ++it) {
            const int pos = 2 * ((int)__builtin_amdgcn_readfirstlane(CW[CW_DONE]) + DN - 4);
            if (rdy * 7 - pos <= RUNWAY) {
                unsigned v = call;
                const char* a = fp + (rdy < 7 ? rdy : 6) * 128;
                if (lane < 16) asm volatile("global_load_dword %0, %1, off sc0 sc1\n\ts_waitcnt vmcnt(0)" : "=v"(v) : "v"(a) : "memory");
                unsigned m = (unsigned)__ballot(self | ((int)(v - call) >= 0)) & 0xffffu;
                if (rdy == 7) m >>= 8;
                const int add = (m & 0xffu) != 0xffu ? 0 : (rdy < 7 && m == 0xffffu ? 2 : 1);
                if (add) {
                    rdy += add;
                    CW[CW_READY] = (unsigned)rdy * 7u;
                    if (rdy >= 8) break;
                    continue;
                }
            }
            if ((it & 15) == 15) {
                const unsigned tv = __hip_atomic_load(tmo, __ATOMIC_RELAXED, __HIP_MEMORY_SCOPE_AGENT);
                if (__builtin_amdgcn_readfirstlane(tv) == call || rt() - t0 > DEADLINE_TICKS) {
                    if (lane == 0) __hip_atomic_store(tmo, call, __ATOMIC_RELAXED, __HIP_MEMORY_SCOPE_AGENT);
                    CW[CW_POISON] = 1u;
                    wait_lgkm0();
                    CW[CW_READY] = 56u;
                    break;
                }
            }
            __builtin_amdgcn_s_sleep(PSLEEP);
        }
    } else {
        const int mh = wave & 1, ks = wave >> 1;
        const int g = lane >> 4, qq = (lane & 15) >> 1;
        unsigned baddr[8];
        {
            const unsigned base_l = (unsigned)(ks * (NS * SLOT) + (16 * g + qq) * 128 + (lane & 1) * 8);
            const int fl = ((qq >> 1) & 3) | ((g & 1) << 2);
#pragma unroll
            for (int nt = 0; nt < 8; ++nt) baddr[nt] = base_l + (unsigned)((nt ^ fl) << 4);
        }
        const __amdgpu_buffer_rsrc_t rsX = __builtin_amdgcn_make_buffer_rsrc((void*)P.x, 0, KTOT * KC, 0x00020000);
        const __amdgpu_buffer_rsrc_t rsA = __builtin_amdgcn_make_buffer_rsrc((void*)wsl, 0, 2 * A_PAR_BYTES, 0x00020000);
        v4i Ab[8][2];
#pragma unroll
        for (int u = 4; u < 8; ++u) { Ab[u][0] = (v4i){0, 0, 0, 0}; Ab[u][1] = (v4i){0, 0, 0, 0}; }
        {
            const unsigned voL = (unsigned)(lane & 15) * KC + (unsigned)g * 16u;
            const unsigned so = (unsigned)(rank * MLOC + mt * 64 + mh * 32) * KC + (unsigned)ks * 64u;
#pragma unroll
            for (int u = 0; u < 4; ++u) {
                Ab[u][0] = __builtin_bit_cast(v4i, __builtin_amdgcn_raw_buffer_load_b128(rsX, voL, so + u * 128u, 0));
                Ab[u][1] = __builtin_bit_cast(v4i, __builtin_amdgcn_raw_buffer_load_b128(rsX, voL, so + u * 128u + 16u * KC, 0));
            }
        }
        unsigned call = __hip_atomic_load(ctrp, __ATOMIC_RELAXED, __HIP_MEMORY_SCOPE_AGENT);
        sxv = P.sx[0]; swv = P.sw[0];
        __builtin_amdgcn_s_barrier();
        const unsigned voA = (unsigned)lane * 16u;
        unsigned soA = 0;
        constexpr unsigned OOB = 0x40000000u;

        v4i acc[2][8];
#pragma unroll
        for (int nt = 0; nt < 8; ++nt) { acc[0][nt] = (v4i){0, 0, 0, 0}; acc[1][nt] = (v4i){0, 0, 0, 0}; }

        unsigned full_seen = 0, ready_seen = 0;
        auto need_full = [&](unsigned n) {
            if (__builtin_expect(full_seen < n, 0)) {
                do { full_seen = __builtin_amdgcn_readfirstlane(CW[CW_FULL + ks]); } while (full_seen < n);
            }
            asm volatile("" ::: "memory");
        };
        auto need_ready = [&](unsigned n) {
            if (__builtin_expect(ready_seen < n, 0)) {
                do { ready_seen = __builtin_amdgcn_readfirstlane(CW[CW_READY]); } while (ready_seen < n);
            }
            asm volatile("" ::: "memory");
        };
        v4i Bf[2][8];
#define MMA(i) do { acc[0][i] = __builtin_amdgcn_mfma_i32_16x16x64_i8(Bf[u & 1][i], Ab[u][0], acc[0][i], 0, 0, 0); \
                    acc[1][i] = __builtin_amdgcn_mfma_i32_16x16x64_i8(Bf[u & 1][i], Ab[u][1], acc[1][i], 0, 0, 0); } while (0)
#define SB __builtin_amdgcn_sched_barrier(0)
#define KEEP(x) asm volatile("" :: "v"(x))
#define RDX(buf, slot, r) do { \
            const v2i x_ = __builtin_amdgcn_ds_read_tr8_b64_v2i32((LAS v2i*)(L + baddr[(r) >> 1] + (slot) * SLOT + ((r) & 1) * 1024)); \
            Bf[buf][(r) >> 1][((r) & 1) * 2] = x_.x; Bf[buf][(r) >> 1][((r) & 1) * 2 + 1] = x_.y; } while (0)
#define RD(r) do { if (((r) & 1) == 0) KEEP(Bf[(u + 1) & 1][(r) >> 1]); RDX((u + 1) & 1, (u + 1) & 7, r); } while (0)
#define LDA(dst, off) do { dst = __builtin_bit_cast(v4i, __builtin_amdgcn_raw_buffer_load_b128(rsA, voA, (off), A_AUX)); } while (0)
        need_full(1);
        if (wave == 0) { STAMP(6); }
#pragma unroll
        for (int r = 0; r < 16; ++r) RDX(0, 0, r);
#pragma unroll
        for (int i = 0; i < 8; ++i) Bf[1][i] = (v4i){0, 0, 0, 0};
        need_full(2);
        unsigned jp = (unsigned)ks + 2u * (DN - 4);
        unsigned donev = 1;
        volatile LAS unsigned* donep = CW + CW_DONE + wave;
#define AWAIT do {} while (0)
        unsigned so2 = OOB;
#define BLOCK(PF) do { \
            AWAIT; \
            SB; MMA(0); SB; RD(0); RD(1); RD(2); \
            SB; MMA(1); SB; \
            unsigned fv = 0, rv = 0; \
            if ((u & 1) == 0) {       \
                asm volatile("s_waitcnt lgkmcnt(3)" ::: "memory");     \
                *donep = donev; \
            } \
            RD(3); RD(4); RD(5); \
            if ((u & 1) == 0) { fv = CW[CW_FULL + ks]; rv = CW[CW_READY]; } \
            SB; MMA(2); SB; RD(6); RD(7); RD(8); \
            SB; MMA(3); SB; RD(9); RD(10); RD(11); \
            SB; MMA(4); SB; RD(12); RD(13); RD(14); \
            unsigned so = so2, rneed = 0; \
            if ((u & 1) == 0) { \
                const bool live = jp - 2u * DN < 56u - 2u * DN, live2 = jp + 2u - 2u * DN < 56u - 2u * DN; \
                so = live ? soA + jp * 1024u : OOB; \
                so2 = live2 ? soA + jp * 1024u + 2048u : OOB; \
                rneed = live2 ? jp + 3u : (live ? jp + 1u : 0u); \
            } \
            SB; MMA(5); SB; RD(15); \
            if ((u & 1) == 0) { \
                full_seen = __builtin_amdgcn_readfirstlane(fv); \
                ready_seen = __builtin_amdgcn_readfirstlane(rv); \
                need_ready(rneed); \
            } \
            SB; MMA(6); SB; \
            if (PF) { LDA(Ab[(u + DN) & 7][0], so); LDA(Ab[(u + DN) & 7][1], so + 65536u); } \
            if ((u & 1) == 0) { need_full(donev + 3u > 32u ? 32u : donev + 3u); donev += 2u; jp += 4u; } \
            SB; MMA(7); SB; } while (0)
#pragma unroll 1
        for (int it = 0; it < 4 - PEEL; ++it) {
#pragma unroll
            for (int u = 0; u < 8; ++u) {
                if (u == 4 && it == 0) {
                    if (wave == 0) STAMP(3);
                    call = __builtin_amdgcn_readfirstlane(call) + 1u;
                    if (lane == 0) __hip_atomic_store(ctrp, call, __ATOMIC_RELAXED, __HIP_MEMORY_SCOPE_AGENT);
                    soA = (call & 1u) * A_PAR_BYTES + (unsigned)(mt * 4 + mh * 2) * 65536u;
#pragma unroll
                    for (int v = 0; v < DN; ++v) {
                        const unsigned j = (unsigned)(2 * v + ks);
                        need_ready(j + 1u);
                        LDA(Ab[(4 + v) & 7][0], soA + j * 1024u);
                        LDA(Ab[(4 + v) & 7][1], soA + j * 1024u + 65536u);
                    }
                }
                BLOCK(true);
            }
        }
#undef BLOCK
#undef RD
#undef RDX
#undef SB
#undef LDA
#undef KEEP
#undef MMA
        wait_lgkm0();
        *donep = 32u;
        if (wave == 0) { STAMP(4); CSTAMP(31); }
        unsigned d;
        do { d = __builtin_amdgcn_readfirstlane(CW[CW_DONE + (wave ^ 1)]); } while (d < 32u);
        asm volatile("" ::: "memory");
        LAS unsigned char* xs = L + ks * (NS * SLOT) + (mh * 32 + (lane & 15)) * XROW + g * 16;
#pragma unroll
        for (int f = 0; f < 2; ++f)
#pragma unroll
            for (int nt = 0; nt < 8; ++nt) *(LAS v4i*)(xs + f * 16 * XROW + nt * 64) = acc[f][nt];
        wait_lgkm0();
        CW[CW_XRDY + wave] = 1u;
        if (wave == 0) STAMP(2);
        if (wave == 2) STAMP(24);
    }

    {
        const float scale0 = sxv * swv;
        const int mhw = wave >> 2;
        const int r0 = wave * 8 + (lane >> 5);
        const LAS unsigned char* xr = L + r0 * XROW + (lane & 31) * 16;
        const __amdgpu_buffer_rsrc_t rsO = __builtin_amdgcn_make_buffer_rsrc((void*)P.out, 0, MLOC * NTOT * 4, 0x00020000);
        const unsigned voO = (unsigned)((mt * 64 + r0) * NTOT + n0 + (lane & 31) * 4) * 4u;
#pragma unroll 1
        for (int pass = (DRY && wave >= 4) ? 0 : 1; pass < 2; ++pass) {
            while (pass) {
                const unsigned a0 = CW[CW_XRDY + mhw], a1 = CW[CW_XRDY + 2 + mhw];
                if (__builtin_amdgcn_readfirstlane(a0 & a1)) break;
                if (wave >= 4) __builtin_amdgcn_s_sleep(2);
            }
            asm volatile("" ::: "memory");
            if (wave == 0) STAMP(7);
            const float scale = __builtin_amdgcn_readfirstlane(CW[CW_POISON]) != 0u ? __builtin_nanf("") : scale0;
#pragma unroll
            for (int i = 0; i < 4; ++i) {
                const v4i p0 = *(const LAS v4i*)(xr + i * 2 * XROW), p1 = *(const LAS v4i*)(xr + i * 2 * XROW + NS * SLOT);
                v4f r;
#pragma unroll
                for (int e = 0; e < 4; ++e) {
                    const float y = (float)(p0[e] + p1[e]) * scale;
                    const float z = __builtin_amdgcn_fmed3f(y * -1.44269504088896341f, -86.5617024533378f, 86.5617024533378f);
                    r[e] = y * __builtin_amdgcn_rcpf(1.0f + __builtin_amdgcn_exp2f(z));
                }
                __builtin_amdgcn_raw_buffer_store_b128(__builtin_bit_cast(v4u, r), rsO, voO + (pass ? 0u : 0x40000000u) + (unsigned)i * 2u * NTOT * 4u, 0, 0);
            }
        }
    }
    if (wave == 0) STAMP(5);
    if (wave == 7) STAMP(25);
}

extern "C" void kernel_launch(int rank, int nranks, ncclComm_t comm,
                              hipStream_t stream,
                              void* const* d_in, const int* in_sizes, int n_in,
                              void* d_out, int out_size,
                              const int* in_dtypes, int out_dtype,
                              void* const* d_out_all,
                              void* const* d_ws_all, size_t ws_bytes) {
    if (nranks != NR || (unsigned)rank >= (unsigned)NR || ws_bytes < WS_NEEDED || n_in < 4 ||
        out_size != MLOC * NTOT || in_sizes[0] != KTOT * KC || in_sizes[1] != KTOT * NTOT) {
        fprintf(stderr, "kernel_launch: unexpected problem geometry\n");
        return;
    }
    Params P;
    P.x = (const int8_t*)d_in[0]; P.w = (const int8_t*)d_in[1];
    P.sx = (const float*)d_in[2]; P.sw = (const float*)d_in[3];
    P.out = (float*)d_out;
    for (int p = 0; p < NR; ++p) P.ws[p] = (char*)d_ws_all[p];
    P.rank = rank; P.wsl = (char*)d_ws_all[rank];
    a2a_gemm_kernel<<<dim3(256), dim3(512), 0, stream>>>(P);
    HIP_CHECK(hipGetLastError());
}
```
